```python
import numpy as np

kernel_name = "gpu_ag_gemm_m8192_k8192_n8192_f32_none"


ABSMAX_RTOL = 0.001

COMPUTE_DTYPE = "float32"
EPILOGUE = "none"
IN_QUANT = None
OUT_QUANT = None
IN_DTYPES = {"x": "float32", "w_mat": "float32"}
WORLD_SIZE = 8

def setup_inputs():

    x = np.random.randn(8192, 8192).astype(np.float32)
    w_mat = (np.random.randn(8192, 8192) / np.sqrt(8192)).astype(np.float32)
    return {"x": x, "w_mat": w_mat}

def _epilogue(y):
    return y

def reference(x, w_mat):
    y = (x @ w_mat).astype(np.float32)
    return _epilogue(y)
```

```cpp
#include <hip/hip_runtime.h>
#include <rccl/rccl.h>
#include <cstdio>
#include <cstring>

#define HIP_CHECK(expr) do { hipError_t _e = (expr); if (_e != hipSuccess) { \
    fprintf(stderr, "HIP error %s at %s:%d\n", hipGetErrorString(_e), __FILE__, __LINE__); \
    return; } } while (0)

#define NPL 18
#define CVW 2
#define CVORD 0x7610FDB95432ECA8ull
#define FSTR 128u
#define PSLEEP 2
#define PSLOW 100
#define PFROM 1
#define ISLEEP 2
#define XSL 4
#define XNA 8
#define LSLEEP 2
#define HFROM 1
#define EFROM 99
#define HSLEEP 1
#define XEARLY 48
#define PP 4
#define EDELAY 8
#define OLAZY 16
#define XR 2
namespace {
#define NCOMM 56
#define NPUSH 3
__constant__ unsigned TAB_LEN[8] = {184, 183, 183, 130, 131, 131, 131, 0};
__constant__ unsigned TAB_JOB[256] = {
    0,1,2,3,4,5,6,7,8,9,10,11,12,13,14,15,
    16,17,18,19,20,21,22,23,24,25,26,27,28,29,30,31,
    32,33,34,35,36,37,38,39,40,41,42,43,44,45,46,47,
    48,49,50,51,52,53,54,55,137889804,137955341,138020878,138086415,138151980,138217517,138283054,138348591,
    222038540,222104077,222169614,222235151,255593004,255658541,255724078,255789615,171444492,171510029,171575566,171641103,171706668,171772205,171837742,171903279,
    490474252,490539789,490605326,490670863,524028716,524094253,524159790,524225327,139986964,140052501,140118038,140183575,140249164,140314701,140380238,140445775,
    224135700,224201237,224266774,224332311,257690188,257755725,257821262,257886799,173541652,173607189,173672726,173738263,173803852,173869389,173934926,174000463,
    492571412,492636949,492702486,492768023,526125900,526191437,526256974,526322511,142084124,142149661,142215198,142280735,142346348,142411885,142477422,142542959,
    226232860,226298397,226363934,226429471,259787372,259852909,259918446,259983983,175638812,175704349,175769886,175835423,175901036,175966573,176032110,176097647,
    494668572,494734109,494799646,494865183,528223084,528288621,528354158,528419695,221775924,221841461,221906998,221972535,255330388,255395925,255461462,255526999,
    490211892,490277429,490342966,490408503,523766356,523831893,523897430,523962967,223873332,223938869,224004406,224069943,257427796,257493333,257558870,257624407,
    492309300,492374837,492440374,492505911,525863764,525929301,525994838,526060375,225969152,226034690,226100232,226165770,259523600,259589138,259654680,259720218,
    494404609,494470147,494535689,494601227,527959057,528024595,528090137,528155675,1064,1099,1074,1137,1184,1219,1254,1157,
    1065,1128,1075,1138,1185,1248,1220,1158,1066,1129,1104,1139,1186,1249,1221,1159,
    1067,1130,1105,1152,1187,1250,1223,1140,1096,1131,1106,1153,1216,1251,1188,1141,
    1097,1072,1107,1154,1217,1252,1190,1142,1098,1073,1136,1155,1218,1253,1191,1143,
};
__constant__ unsigned TAB_Q[1344] = {
    260256,260161,260193,260225,260257,260162,260194,260226,260258,260163,260195,260227,260259,260164,260196,260228,260260,260165,260197,260229,260261,260166,260198,260230,
    260262,260167,260199,260168,260200,260231,260263,260169,260201,260232,260264,260170,260202,260233,260265,260171,260203,260234,260266,260172,260204,260235,260267,260173,
    260205,260236,260268,260174,260206,260237,260269,260175,260207,260238,260270,260176,260208,260239,260271,260177,260209,260178,260210,260240,260272,260179,260211,260241,
    260273,260180,260212,260242,260274,260181,260213,260243,260275,260182,260214,260244,260276,260183,260215,260245,260277,260184,260216,260246,260278,260185,260217,260247,
    260279,260186,260218,260248,260280,260187,260219,260188,260249,260220,260281,260189,260221,260250,260282,260190,260222,260251,260283,260191,260223,260252,260284,260288,
    260289,260290,260291,260297,260253,260298,260285,260299,260254,260300,260286,260301,260255,260302,260287,260303,260304,260305,260306,260307,260308,260309,260310,260311,
    260312,260313,260314,260315,260316,260317,260318,260319,260320,260321,260322,260323,260324,260325,260326,260327,260328,260329,260330,260331,260332,260333,260334,260335,
    260336,260337,260338,260339,260340,260341,260342,260343,260344,260345,260346,260347,260348,260349,260350,260351,0,0,0,0,0,0,0,0,
    260256,260161,260193,260225,260257,260162,260194,260226,260258,260163,260195,260227,260259,260164,260196,260228,260260,260165,260197,260229,260261,260166,260198,260230,
    260262,260167,260199,260168,260200,260231,260263,260169,260201,260232,260264,260170,260202,260233,260265,260171,260203,260234,260266,260172,260204,260235,260267,260173,
    260205,260236,260268,260174,260206,260237,260269,260175,260207,260238,260270,260176,260208,260239,260271,260177,260209,260178,260210,260240,260272,260179,260211,260241,
    260273,260180,260212,260242,260274,260181,260213,260243,260275,260182,260214,260244,260276,260183,260215,260245,260277,260184,260216,260246,260278,260185,260217,260247,
    260279,260186,260218,260248,260280,260187,260219,260188,260249,260220,260281,260189,260221,260250,260282,260190,260222,260251,260283,260191,260223,260252,260284,260288,
    260289,260290,260291,260253,260298,260285,260299,260254,260300,260286,260301,260255,260302,260287,260303,260304,260305,260306,260307,260308,260309,260310,260311,260312,
    260313,260314,260315,260316,260317,260318,260319,260320,260321,260322,260323,260324,260325,260326,260327,260328,260329,260330,260331,260332,260333,260334,260335,260336,
    260337,260338,260339,260340,260341,260342,260343,260344,260345,260346,260347,260348,260349,260350,260351,0,0,0,0,0,0,0,0,0,
    260256,260161,260193,260225,260257,260162,260194,260226,260258,260163,260195,260227,260259,260164,260196,260228,260260,260165,260197,260229,260261,260166,260198,260230,
    260262,260167,260199,260168,260200,260231,260263,260169,260201,260232,260264,260170,260202,260233,260265,260171,260203,260234,260266,260172,260204,260235,260267,260173,
    260205,260236,260268,260174,260206,260237,260269,260175,260207,260238,260270,260176,260208,260239,260271,260177,260209,260178,260210,260240,260272,260179,260211,260241,
    260273,260180,260212,260242,260274,260181,260213,260243,260275,260182,260214,260244,260276,260183,260215,260245,260277,260184,260216,260246,260278,260185,260217,260247,
    260279,260186,260218,260248,260280,260187,260219,260188,260249,260220,260281,260189,260221,260250,260282,260190,260222,260251,260283,260191,260223,260252,260284,260288,
    260289,260290,260291,260253,260298,260285,260299,260254,260300,260286,260301,260255,260302,260287,260303,260304,260305,260306,260307,260308,260309,260310,260311,260312,
    260313,260314,260315,260316,260317,260318,260319,260320,260321,260322,260323,260324,260325,260326,260327,260328,260329,260330,260331,260332,260333,260334,260335,260336,
    260337,260338,260339,260340,260341,260342,260343,260344,260345,260346,260347,260348,260349,260350,260351,0,0,0,0,0,0,0,0,0,
    260128,261121,261153,260097,260129,261122,261154,260098,260130,261123,261155,260099,260131,261124,261156,261125,260100,261157,260132,261126,261158,260101,260133,261127,
    196,261159,260102,260134,261128,261160,260103,260135,261129,261161,260104,260136,261130,261162,260105,260137,261131,261163,260106,260138,261132,261164,261133,261165,
    260107,260139,261134,261166,260108,260140,261135,261167,260109,260141,261136,261168,260110,260142,261137,261169,260111,260143,261138,261170,260112,260144,261139,261171,
    261140,261172,260113,260145,261141,261173,260114,260146,261142,261174,260115,260147,261143,261175,260116,260148,261144,261176,260117,260149,261145,261177,260118,260150,
    261146,261178,261147,261179,260119,260151,261148,261180,260120,260152,261149,261181,260121,260153,261150,261182,260122,260154,261151,261183,260123,260155,260124,260156,
    260125,260157,260126,260158,260127,260159,261061,260806,260551,261064,0,0,0,0,0,0,0,0,0,0,0,0,0,0,
    0,0,0,0,0,0,0,0,0,0,0,0,0,0,0,0,0,0,0,0,0,0,0,0,
    0,0,0,0,0,0,0,0,0,0,0,0,0,0,0,0,0,0,0,0,0,0,0,0,
    260128,261121,261153,260097,260129,261122,261154,260098,260130,261123,261155,260099,260131,261124,261156,261125,260100,261157,260132,261126,261158,260101,260133,261127,
    6343,261159,260102,260134,261128,261160,260103,260135,10441,261129,261161,260104,260136,261130,261162,260105,260137,261131,261163,260106,260138,261132,261164,261133,
    261165,260107,260139,261134,261166,260108,260140,261135,261167,260109,260141,261136,261168,260110,260142,261137,261169,260111,260143,261138,261170,260112,260144,261139,
    261171,261140,261172,260113,260145,261141,261173,260114,260146,261142,261174,260115,260147,261143,261175,260116,260148,261144,261176,260117,260149,261145,261177,260118,
    260150,261146,261178,261147,261179,260119,260151,261148,261180,260120,260152,261149,261181,260121,260153,261150,261182,260122,260154,261151,261183,260123,260155,260124,
    260156,260125,260157,260126,260158,260127,260159,260548,261062,260807,260809,0,0,0,0,0,0,0,0,0,0,0,0,0,
    0,0,0,0,0,0,0,0,0,0,0,0,0,0,0,0,0,0,0,0,0,0,0,0,
    0,0,0,0,0,0,0,0,0,0,0,0,0,0,0,0,0,0,0,0,0,0,0,0,
    260128,261121,261153,260097,260129,261122,261154,260098,260130,261123,261155,260099,260131,261124,261156,261125,260100,261157,260132,261126,261158,260101,260133,261127,
    4294,261159,260102,260134,261128,261160,260103,260135,261129,261161,260104,260136,261130,261162,260105,260137,261131,261163,260106,260138,261132,261164,261133,261165,
    260107,260139,261134,261166,260108,260140,261135,261167,260109,260141,261136,261168,260110,260142,261137,261169,260111,260143,261138,261170,260112,260144,261139,261171,
    261140,261172,260113,260145,261141,261173,260114,260146,261142,261174,260115,260147,261143,261175,260116,260148,261144,261176,260117,260149,261145,261177,260118,260150,
    261146,261178,261147,261179,260119,260151,261148,261180,260120,260152,261149,261181,260121,260153,261150,261182,260122,260154,261151,261183,260123,260155,260124,260156,
    260125,260157,260126,260158,260127,260159,260804,260549,261063,260552,261065,0,0,0,0,0,0,0,0,0,0,0,0,0,
    0,0,0,0,0,0,0,0,0,0,0,0,0,0,0,0,0,0,0,0,0,0,0,0,
    0,0,0,0,0,0,0,0,0,0,0,0,0,0,0,0,0,0,0,0,0,0,0,0,
    260128,261121,261153,260097,260129,261122,261154,260098,260130,261123,261155,260099,260131,261124,261156,261125,260100,261157,260132,261126,261158,260101,260133,261127,
    2245,261159,260102,260134,261128,261160,260103,260135,8392,261129,261161,260104,260136,261130,261162,260105,260137,261131,261163,260106,260138,261132,261164,261133,
    261165,260107,260139,261134,261166,260108,260140,261135,261167,260109,260141,261136,261168,260110,260142,261137,261169,260111,260143,261138,261170,260112,260144,261139,
    261171,261140,261172,260113,260145,261141,261173,260114,260146,261142,261174,260115,260147,261143,261175,260116,260148,261144,261176,260117,260149,261145,261177,260118,
    260150,261146,261178,261147,261179,260119,260151,261148,261180,260120,260152,261149,261181,260121,260153,261150,261182,260122,260154,261151,261183,260123,260155,260124,
    260156,260125,260157,260126,260158,260127,260159,261060,260805,260550,260808,0,0,0,0,0,0,0,0,0,0,0,0,0,
    0,0,0,0,0,0,0,0,0,0,0,0,0,0,0,0,0,0,0,0,0,0,0,0,
    0,0,0,0,0,0,0,0,0,0,0,0,0,0,0,0,0,0,0,0,0,0,0,0,
};
__constant__ unsigned char TAB_NT[4144] = {
    255,255,255,255,255,255,255,255,255,255,255,3,3,3,3,255,255,255,255,7,7,7,7,255,255,255,255,11,11,11,11,255,255,255,255,16,16,16,16,255,255,255,255,21,21,21,21,255,
    255,255,255,26,26,26,26,255,255,255,255,30,30,30,30,255,255,255,255,34,35,34,35,255,255,255,255,38,39,38,39,255,255,255,255,42,43,42,43,255,255,255,255,48,49,48,49,255,
    255,255,255,52,53,52,53,255,255,255,255,56,57,56,57,255,255,255,255,60,61,60,61,255,255,255,255,64,65,64,65,255,255,255,255,68,69,68,69,255,255,255,255,74,75,74,75,255,
    255,255,255,78,79,78,79,255,255,255,255,82,83,82,83,255,255,255,255,86,87,86,87,255,255,255,255,90,91,90,91,255,255,255,255,94,95,94,95,255,255,255,255,100,101,100,101,255,
    255,255,255,104,105,104,105,255,255,255,255,108,109,108,109,255,255,255,255,112,113,112,113,255,255,255,255,116,117,116,117,255,255,255,255,118,119,118,119,255,255,255,255,120,121,120,121,255,
    255,255,255,122,123,122,123,255,255,255,255,124,125,124,125,255,255,255,255,0,0,0,0,255,255,255,255,4,4,4,4,255,255,255,255,8,8,8,8,255,255,255,255,12,12,12,12,255,
    255,255,255,18,18,18,18,255,255,255,255,22,22,22,22,255,255,255,255,27,27,27,27,255,255,255,255,31,31,31,31,255,255,255,255,35,36,35,36,255,255,255,255,39,40,39,40,255,
    255,255,255,43,44,43,44,255,255,255,255,49,50,49,50,255,255,255,255,53,54,53,54,255,255,255,255,57,58,57,58,255,255,255,255,61,62,61,62,255,255,255,255,65,66,65,66,255,
    255,255,255,69,70,69,70,255,255,255,255,75,76,75,76,255,255,255,255,79,80,79,80,255,255,255,255,83,84,83,84,255,255,255,255,87,88,87,88,255,255,255,255,91,92,91,92,255,
    255,255,255,95,96,95,96,255,255,255,255,101,102,101,102,255,255,255,255,105,106,105,106,255,255,255,255,109,110,109,110,255,255,255,255,113,114,113,114,255,255,255,255,117,118,117,118,255,
    255,255,255,119,120,119,120,255,255,255,255,121,122,121,122,255,255,255,255,123,124,123,124,255,255,255,255,125,126,125,126,255,255,255,255,255,255,255,255,255,1,1,1,255,255,255,255,255,
    5,5,5,255,255,255,255,255,9,9,9,255,255,255,255,255,13,13,13,255,255,255,255,255,17,17,17,255,255,255,255,255,21,21,21,255,255,255,255,255,25,25,25,255,255,255,255,255,
    27,27,27,255,255,255,255,255,31,31,31,255,255,255,255,255,35,35,35,255,255,255,255,255,39,39,39,255,255,255,255,255,43,43,43,255,255,255,255,255,47,47,47,255,255,255,255,255,
    51,51,51,255,255,255,255,255,55,55,55,255,255,255,255,255,59,59,59,255,255,255,255,255,63,63,63,255,255,255,255,255,65,65,65,255,255,255,255,255,69,69,69,255,255,255,255,255,
    73,73,73,255,255,255,255,255,77,77,77,255,255,255,255,255,81,81,81,255,255,255,255,255,85,85,85,255,255,255,255,255,89,89,89,255,255,255,255,255,93,93,93,255,255,255,255,255,
    97,97,97,255,255,255,255,255,101,101,101,255,255,255,255,255,103,103,103,255,255,255,255,255,107,107,107,255,255,255,255,255,111,111,111,255,255,255,255,255,115,115,115,255,255,255,255,255,
    255,255,255,255,255,255,255,255,2,2,2,255,255,255,255,255,6,6,6,255,255,255,255,255,10,10,10,255,255,255,255,255,14,14,14,255,255,255,255,255,18,18,18,255,255,255,255,255,
    22,22,22,255,255,255,255,255,26,26,26,255,255,255,255,255,28,28,28,255,255,255,255,255,32,32,32,255,255,255,255,255,36,36,36,255,255,255,255,255,40,40,40,255,255,255,255,255,
    44,44,44,255,255,255,255,255,48,48,48,255,255,255,255,255,52,52,52,255,255,255,255,255,56,56,56,255,255,255,255,255,60,60,60,255,255,255,255,255,64,64,64,255,255,255,255,255,
    66,66,66,255,255,255,255,255,70,70,70,255,255,255,255,255,74,74,74,255,255,255,255,255,78,78,78,255,255,255,255,255,82,82,82,255,255,255,255,255,86,86,86,255,255,255,255,255,
    90,90,90,255,255,255,255,255,94,94,94,255,255,255,255,255,98,98,98,255,255,255,255,255,102,102,102,255,255,255,255,255,105,105,105,255,255,255,255,255,108,108,108,255,255,255,255,255,
    112,112,112,255,255,255,255,255,116,116,116,255,255,255,255,255,255,255,255,255,255,255,255,255,3,3,3,255,255,255,255,255,7,7,7,255,255,255,255,255,11,11,11,255,255,255,255,255,
    15,15,15,255,255,255,255,255,19,19,19,255,255,255,255,255,23,23,23,255,255,255,255,255,29,29,29,255,255,255,255,255,33,33,33,255,255,255,255,255,37,37,37,255,255,255,255,255,
    41,41,41,255,255,255,255,255,45,45,45,255,255,255,255,255,49,49,49,255,255,255,255,255,53,53,53,255,255,255,255,255,57,57,57,255,255,255,255,255,61,61,61,255,255,255,255,255,
    67,67,67,255,255,255,255,255,71,71,71,255,255,255,255,255,75,75,75,255,255,255,255,255,79,79,79,255,255,255,255,255,83,83,83,255,255,255,255,255,87,87,87,255,255,255,255,255,
    91,91,91,255,255,255,255,255,95,95,95,255,255,255,255,255,99,99,99,255,255,255,255,255,104,104,104,255,255,255,255,255,109,109,109,255,255,255,255,255,113,113,113,255,255,255,255,255,
    117,117,117,255,255,255,255,255,123,123,124,255,255,255,255,255,127,127,128,255,255,255,255,255,131,131,132,255,255,255,255,255,0,0,0,255,255,255,255,255,4,4,4,255,255,255,255,255,
    8,8,8,255,255,255,255,255,12,12,12,255,255,255,255,255,16,16,16,255,255,255,255,255,20,20,20,255,255,255,255,255,24,24,24,255,255,255,255,255,30,30,30,255,255,255,255,255,
    34,34,34,255,255,255,255,255,38,38,38,255,255,255,255,255,42,42,42,255,255,255,255,255,46,46,46,255,255,255,255,255,50,50,50,255,255,255,255,255,54,54,54,255,255,255,255,255,
    58,58,58,255,255,255,255,255,62,62,62,255,255,255,255,255,68,68,68,255,255,255,255,255,72,72,72,255,255,255,255,255,76,76,76,255,255,255,255,255,80,80,80,255,255,255,255,255,
    84,84,84,255,255,255,255,255,88,88,88,255,255,255,255,255,92,92,92,255,255,255,255,255,96,96,96,255,255,255,255,255,100,100,100,255,255,255,255,255,106,106,106,255,255,255,255,255,
    110,110,110,255,255,255,255,255,114,114,114,255,255,255,255,255,118,118,118,255,255,255,255,255,125,125,126,255,255,255,255,255,129,129,130,255,255,255,255,255,133,133,134,255,255,255,255,255,
    119,119,119,255,255,255,255,255,120,120,120,255,255,255,255,255,121,121,121,255,255,255,255,255,122,122,122,255,255,255,255,255,255,255,255,24,255,255,255,255,255,255,255,255,24,255,255,255,
    255,255,255,255,255,24,255,255,255,255,255,255,255,255,24,255,255,255,255,255,32,255,255,255,255,255,123,255,255,255,32,255,124,124,125,255,255,255,255,255,126,126,127,255,255,255,255,255,
    128,128,129,255,255,255,255,255,130,130,131,255,255,255,255,255,132,132,133,255,255,255,255,255,134,134,135,255,255,255,255,255,135,135,136,255,255,255,255,255,136,136,137,255,255,255,255,255,
    137,137,138,255,255,255,255,255,138,138,139,255,255,255,255,255,139,139,140,255,255,255,255,255,140,140,141,255,255,255,255,255,141,141,142,255,255,255,255,255,142,142,143,255,255,255,255,255,
    143,143,144,255,255,255,255,255,144,144,145,255,255,255,255,255,145,145,146,255,255,255,255,255,146,146,147,255,255,255,255,255,147,147,148,255,255,255,255,255,148,148,149,255,255,255,255,255,
    149,149,150,255,255,255,255,255,150,150,151,255,255,255,255,255,151,151,152,255,255,255,255,255,152,152,153,255,255,255,255,255,153,153,154,255,255,255,255,255,154,154,155,255,255,255,255,255,
    155,155,156,255,255,255,255,255,156,156,157,255,255,255,255,255,157,157,158,255,255,255,255,255,158,158,159,255,255,255,255,255,159,159,160,255,255,255,255,255,160,160,161,255,255,255,255,255,
    161,161,162,255,255,255,255,255,162,162,163,255,255,255,255,255,163,163,164,255,255,255,255,255,164,164,165,255,255,255,255,255,165,165,166,255,255,255,255,255,166,166,167,255,255,255,255,255,
    167,167,168,255,255,255,255,255,168,168,169,255,255,255,255,255,169,169,170,255,255,255,255,255,170,170,171,255,255,255,255,255,171,171,172,255,255,255,255,255,172,172,173,255,255,255,255,255,
    173,173,174,255,255,255,255,255,174,174,175,255,255,255,255,255,175,175,176,255,255,255,255,255,176,176,177,255,255,255,255,255,177,177,178,255,255,255,255,255,178,178,179,255,255,255,255,255,
    179,179,180,255,255,255,255,255,180,180,181,255,255,255,255,255,181,181,182,255,255,255,255,255,182,182,183,255,255,255,255,255,255,255,255,255,255,255,255,255,255,255,255,1,255,255,255,255,
    255,255,255,5,255,255,255,255,255,255,255,9,255,255,255,255,255,255,255,13,255,255,255,255,255,255,255,15,255,255,255,255,255,255,255,19,255,255,255,255,255,255,255,23,255,255,255,255,
    255,255,255,28,255,255,255,255,255,255,255,32,255,255,255,255,255,255,255,36,255,255,255,255,255,255,255,40,255,255,255,255,255,255,255,44,255,255,255,255,255,255,255,46,255,255,255,255,
    255,255,255,50,255,255,255,255,255,255,255,54,255,255,255,255,255,255,255,58,255,255,255,255,255,255,255,62,255,255,255,255,255,255,255,66,255,255,255,255,255,255,255,70,255,255,255,255,
    255,255,255,72,255,255,255,255,255,255,255,76,255,255,255,255,255,255,255,80,255,255,255,255,255,255,255,84,255,255,255,255,255,255,255,88,255,255,255,255,255,255,255,92,255,255,255,255,
    255,255,255,96,255,255,255,255,255,255,255,98,255,255,255,255,255,255,255,102,255,255,255,255,255,255,255,106,255,255,255,255,255,255,255,110,255,255,255,255,255,255,255,114,255,255,255,255,
    255,255,255,255,255,255,255,255,255,255,255,2,255,255,255,255,255,255,255,6,255,255,255,255,255,255,255,10,255,255,255,255,255,255,255,14,255,255,255,255,255,255,255,17,255,255,255,255,
    255,255,255,20,255,255,255,255,255,255,255,25,255,255,255,255,255,255,255,29,255,255,255,255,255,255,255,33,255,255,255,255,255,255,255,37,255,255,255,255,255,255,255,41,255,255,255,255,
    255,255,255,45,255,255,255,255,255,255,255,47,255,255,255,255,255,255,255,51,255,255,255,255,255,255,255,55,255,255,255,255,255,255,255,59,255,255,255,255,255,255,255,63,255,255,255,255,
    255,255,255,67,255,255,255,255,255,255,255,71,255,255,255,255,255,255,255,73,255,255,255,255,255,255,255,77,255,255,255,255,255,255,255,81,255,255,255,255,255,255,255,85,255,255,255,255,
    255,255,255,89,255,255,255,255,255,255,255,93,255,255,255,255,255,255,255,97,255,255,255,255,255,255,255,99,255,255,255,255,255,255,255,103,255,255,255,255,255,255,255,107,255,255,255,255,
    255,255,255,111,255,255,255,255,255,255,255,115,255,255,255,255,255,255,255,255,255,255,255,255,255,255,255,255,1,255,255,255,255,255,255,255,5,255,255,255,255,255,255,255,9,255,255,255,
    255,255,255,255,13,255,255,255,255,255,255,255,15,255,255,255,255,255,255,255,19,255,255,255,255,255,255,255,23,255,255,255,255,255,255,255,28,255,255,255,255,255,255,255,33,255,255,255,
    255,255,255,255,37,255,255,255,255,255,255,255,41,255,255,255,255,255,255,255,45,255,255,255,255,255,255,255,47,255,255,255,255,255,255,255,51,255,255,255,255,255,255,255,55,255,255,255,
    255,255,255,255,59,255,255,255,255,255,255,255,63,255,255,255,255,255,255,255,67,255,255,255,255,255,255,255,71,255,255,255,255,255,255,255,73,255,255,255,255,255,255,255,77,255,255,255,
    255,255,255,255,81,255,255,255,255,255,255,255,85,255,255,255,255,255,255,255,89,255,255,255,255,255,255,255,93,255,255,255,255,255,255,255,97,255,255,255,255,255,255,255,99,255,255,255,
    255,255,255,255,103,255,255,255,255,255,255,255,107,255,255,255,255,255,255,255,111,255,255,255,255,255,255,255,115,255,255,255,255,255,255,255,255,255,255,255,255,255,255,255,2,255,255,255,
    255,255,255,255,6,255,255,255,255,255,255,255,10,255,255,255,255,255,255,255,14,255,255,255,255,255,255,255,17,255,255,255,255,255,255,255,20,255,255,255,255,255,255,255,25,255,255,255,
    255,255,255,255,29,255,255,255,255,255,255,255,34,255,255,255,255,255,255,255,38,255,255,255,255,255,255,255,42,255,255,255,255,255,255,255,46,255,255,255,255,255,255,255,48,255,255,255,
    255,255,255,255,52,255,255,255,255,255,255,255,56,255,255,255,255,255,255,255,60,255,255,255,255,255,255,255,64,255,255,255,255,255,255,255,68,255,255,255,255,255,255,255,72,255,255,255,
    255,255,255,255,74,255,255,255,255,255,255,255,78,255,255,255,255,255,255,255,82,255,255,255,255,255,255,255,86,255,255,255,255,255,255,255,90,255,255,255,255,255,255,255,94,255,255,255,
    255,255,255,255,98,255,255,255,255,255,255,255,100,255,255,255,255,255,255,255,104,255,255,255,255,255,255,255,108,255,255,255,255,255,255,255,112,255,255,255,255,255,255,255,116,255,255,255,
    255,255,255,255,255,255,255,255,255,255,255,255,255,1,255,255,255,255,255,255,255,5,255,255,255,255,255,255,255,9,255,255,255,255,255,255,255,13,255,255,255,255,255,255,255,15,255,255,
    255,255,255,255,255,19,255,255,255,255,255,255,255,23,255,255,255,255,255,255,255,28,255,255,255,255,255,255,255,32,255,255,255,255,255,255,255,36,255,255,255,255,255,255,255,40,255,255,
    255,255,255,255,255,44,255,255,255,255,255,255,255,46,255,255,255,255,255,255,255,50,255,255,255,255,255,255,255,54,255,255,255,255,255,255,255,58,255,255,255,255,255,255,255,62,255,255,
    255,255,255,255,255,66,255,255,255,255,255,255,255,70,255,255,255,255,255,255,255,72,255,255,255,255,255,255,255,76,255,255,255,255,255,255,255,80,255,255,255,255,255,255,255,84,255,255,
    255,255,255,255,255,88,255,255,255,255,255,255,255,92,255,255,255,255,255,255,255,96,255,255,255,255,255,255,255,98,255,255,255,255,255,255,255,102,255,255,255,255,255,255,255,106,255,255,
    255,255,255,255,255,110,255,255,255,255,255,255,255,114,255,255,255,255,255,255,255,255,255,255,255,255,255,255,255,2,255,255,255,255,255,255,255,6,255,255,255,255,255,255,255,10,255,255,
    255,255,255,255,255,14,255,255,255,255,255,255,255,17,255,255,255,255,255,255,255,20,255,255,255,255,255,255,255,25,255,255,255,255,255,255,255,29,255,255,255,255,255,255,255,33,255,255,
    255,255,255,255,255,37,255,255,255,255,255,255,255,41,255,255,255,255,255,255,255,45,255,255,255,255,255,255,255,47,255,255,255,255,255,255,255,51,255,255,255,255,255,255,255,55,255,255,
    255,255,255,255,255,59,255,255,255,255,255,255,255,63,255,255,255,255,255,255,255,67,255,255,255,255,255,255,255,71,255,255,255,255,255,255,255,73,255,255,255,255,255,255,255,77,255,255,
    255,255,255,255,255,81,255,255,255,255,255,255,255,85,255,255,255,255,255,255,255,89,255,255,255,255,255,255,255,93,255,255,255,255,255,255,255,97,255,255,255,255,255,255,255,99,255,255,
    255,255,255,255,255,103,255,255,255,255,255,255,255,107,255,255,255,255,255,255,255,111,255,255,255,255,255,255,255,115,255,255,255,255,255,255,255,255,255,255,255,255,255,255,255,255,1,255,
    255,255,255,255,255,255,5,255,255,255,255,255,255,255,9,255,255,255,255,255,255,255,13,255,255,255,255,255,255,255,15,255,255,255,255,255,255,255,19,255,255,255,255,255,255,255,23,255,
    255,255,255,255,255,255,28,255,255,255,255,255,255,255,33,255,255,255,255,255,255,255,37,255,255,255,255,255,255,255,41,255,255,255,255,255,255,255,45,255,255,255,255,255,255,255,47,255,
    255,255,255,255,255,255,51,255,255,255,255,255,255,255,55,255,255,255,255,255,255,255,59,255,255,255,255,255,255,255,63,255,255,255,255,255,255,255,67,255,255,255,255,255,255,255,71,255,
    255,255,255,255,255,255,73,255,255,255,255,255,255,255,77,255,255,255,255,255,255,255,81,255,255,255,255,255,255,255,85,255,255,255,255,255,255,255,89,255,255,255,255,255,255,255,93,255,
    255,255,255,255,255,255,97,255,255,255,255,255,255,255,99,255,255,255,255,255,255,255,103,255,255,255,255,255,255,255,107,255,255,255,255,255,255,255,111,255,255,255,255,255,255,255,115,255,
    255,255,255,255,255,255,255,255,255,255,255,255,255,255,2,255,255,255,255,255,255,255,6,255,255,255,255,255,255,255,10,255,255,255,255,255,255,255,14,255,255,255,255,255,255,255,17,255,
    255,255,255,255,255,255,20,255,255,255,255,255,255,255,25,255,255,255,255,255,255,255,29,255,255,255,255,255,255,255,34,255,255,255,255,255,255,255,38,255,255,255,255,255,255,255,42,255,
    255,255,255,255,255,255,46,255,255,255,255,255,255,255,48,255,255,255,255,255,255,255,52,255,255,255,255,255,255,255,56,255,255,255,255,255,255,255,60,255,255,255,255,255,255,255,64,255,
    255,255,255,255,255,255,68,255,255,255,255,255,255,255,72,255,255,255,255,255,255,255,74,255,255,255,255,255,255,255,78,255,255,255,255,255,255,255,82,255,255,255,255,255,255,255,86,255,
    255,255,255,255,255,255,90,255,255,255,255,255,255,255,94,255,255,255,255,255,255,255,98,255,255,255,255,255,255,255,100,255,255,255,255,255,255,255,104,255,255,255,255,255,255,255,108,255,
    255,255,255,255,255,255,112,255,255,255,255,255,255,255,116,255,255,255,255,255,127,126,127,255,255,255,255,126,128,127,255,255,255,255,255,127,129,255,128,255,255,255,255,128,255,128,129,255,
    255,255,255,129,130,129,255,255,255,255,255,255,255,130,130,255,
};
}

#define LAS __attribute__((address_space(3)))
#define GAS __attribute__((address_space(1)))
typedef _Float16 f16x8 __attribute__((ext_vector_type(8)));
typedef _Float16 f16x2 __attribute__((ext_vector_type(2)));
typedef float f32x4 __attribute__((ext_vector_type(4)));
typedef unsigned u32x4 __attribute__((ext_vector_type(4)));
typedef GAS unsigned gu32;

#define WAIT_V_(n) asm volatile("s_waitcnt vmcnt(" #n ")" ::: "memory")
#define WAIT_V(n) WAIT_V_(n)
#define WAIT_L(n) asm volatile("s_waitcnt lgkmcnt(" #n ")" ::: "memory")
#define BAR __builtin_amdgcn_s_barrier()
#define SCHED __builtin_amdgcn_sched_barrier(0)

namespace {
constexpr int NR = 8;
constexpr int KDIM = 8192, NLOC = 1024;
constexpr int NKT = 128;
constexpr int NST = 32;
constexpr unsigned HTB = 16384;
#define PADH 0
constexpr unsigned HALFB = NKT * HTB + PADH;
constexpr unsigned CHUNKB = 2 * HALFB;
constexpr unsigned OFF_TMO = 256;
constexpr unsigned OFF_CALL = 49152;
constexpr unsigned OFF_F = 524288;
constexpr unsigned OFF_XF = 14336;
constexpr unsigned OFF_TK = 16384;
constexpr unsigned OFF_RF = 32768, RFP = 0x90000;
constexpr unsigned OFF_X = 1u << 20;
constexpr unsigned OFF_W = OFF_X + 2u * 32u * CHUNKB;
constexpr unsigned OFF_Y = OFF_W + 2u * 8u * CHUNKB;
constexpr unsigned WS_NEED = OFF_Y + 2u * 256u * 65536u;
constexpr unsigned STAGE_BYTES = 8 * HTB;
constexpr unsigned LDS_BYTES = STAGE_BYTES + 64;
constexpr unsigned DEADLINE = 50000000u;
constexpr int NWG = 256, QS = 192, NYB = 60;

__device__ __forceinline__ unsigned cnt_off(unsigned im, unsigned b) { return OFF_F + (im * 2 + (b >> 5)) * FSTR + (b & 31u) * 4; }
struct Params {
    const float* x; const float* w;
    char* me; float* out;
    int rank, pad;
    char* lw[7];
};

__device__ __forceinline__ int lds_byte(int r, int c) { const int st = (r >> 4) * 2 + (c >> 5), rr = r & 15, cc = c & 31, ob = rr * 64 + cc * 2; return st * 1024 + (ob ^ (((ob >> 9) & 1) << 5)); }
__device__ __forceinline__ unsigned now() { return (unsigned)__builtin_amdgcn_s_memrealtime(); }
__device__ __forceinline__ unsigned rfl(unsigned v) { return (unsigned)__builtin_amdgcn_readfirstlane((int)v); }
__device__ __forceinline__ unsigned rdl(unsigned v, unsigned l) { return (unsigned)__builtin_amdgcn_readlane((int)v, (int)l); }
__device__ __forceinline__ char* uni(const void* p) {
    const unsigned long long a = (unsigned long long)p;
    return (char*)(((unsigned long long)rfl((unsigned)(a >> 32)) << 32) | rfl((unsigned)a));
}
__device__ __forceinline__ __amdgpu_buffer_rsrc_t mk_rsrc(const void* p, unsigned bytes) {
    return __builtin_amdgcn_make_buffer_rsrc((void*)uni(p), 0, (int)rfl(bytes), 0x00020000);
}
constexpr unsigned OOB = 0xf0000000u;
constexpr int AUX_WT = 17;
__device__ __forceinline__ void st16(u32x4 v, __amdgpu_buffer_rsrc_t rs, unsigned vo, unsigned so) {
    asm volatile("buffer_store_dwordx4 %0, %1, %2, %3 offen sc0 sc1\n\ts_nop 1" :: "v"(v), "v"(vo), "s"(rs), "s"(so) : "memory");
}
__device__ __forceinline__ void st16x4(u32x4 a, u32x4 b, u32x4 c, u32x4 d, __amdgpu_buffer_rsrc_t rs, unsigned vo, unsigned so, unsigned pitch) {
    asm volatile("s_nop 4\n\tbuffer_store_dwordx4 %0, %4, %5, %6 offen sc0 sc1\n\tbuffer_store_dwordx4 %1, %4, %5, %7 offen sc0 sc1\n\t"
                 "buffer_store_dwordx4 %2, %4, %5, %8 offen sc0 sc1\n\tbuffer_store_dwordx4 %3, %4, %5, %9 offen sc0 sc1\n\ts_nop 1"
                 :: "v"(a), "v"(b), "v"(c), "v"(d), "v"(vo), "s"(rs), "s"(so), "s"(so + pitch), "s"(so + 2 * pitch), "s"(so + 3 * pitch) : "memory");
}
__device__ __forceinline__ unsigned pk2(float a, float b) { f16x2 h; h.x = (_Float16)a; h.y = (_Float16)b; return __builtin_bit_cast(unsigned, h); }
__device__ __forceinline__ u32x4 cvt8(f32x4 a, f32x4 b) { u32x4 r; r.x = pk2(a.x, a.y); r.y = pk2(a.z, a.w); r.z = pk2(b.x, b.y); r.w = pk2(b.z, b.w); return r; }
__device__ __forceinline__ bool ge(unsigned v, unsigned call) { return (int)(v - call) >= 0; }
__device__ __forceinline__ void st_sys(char* p, unsigned v) { __hip_atomic_store((gu32*)p, v, __ATOMIC_RELAXED, __HIP_MEMORY_SCOPE_SYSTEM); }
__device__ __forceinline__ unsigned ld_sys(const char* p) {
    unsigned v;
    asm volatile("global_load_dword %0, %1, off sc0 sc1\n\ts_waitcnt vmcnt(0)" : "=&v"(v) : "v"(p) : "memory");
    return v;
}
__device__ __forceinline__ unsigned add1(const char* p, unsigned n) {
    unsigned o;
    asm volatile("s_mov_b64 exec, 1\n\ts_nop 3\n\tglobal_atomic_add %0, %1, %2, %3 sc0 sc1\n\ts_mov_b64 exec, -1" : "=&v"(o) : "v"(0u), "v"(n), "s"(uni(p)) : "memory");
    return o;
}
__device__ __forceinline__ void add0(const char* p, unsigned n = 1u) {
    asm volatile("s_mov_b64 exec, 1\n\ts_nop 3\n\tglobal_atomic_add %0, %1, %2 sc1\n\ts_mov_b64 exec, -1" :: "v"(0u), "v"(n), "s"(uni(p)) : "memory");
}
__device__ __forceinline__ unsigned got(unsigned v) { unsigned s; asm volatile("v_readfirstlane_b32 %0, %1" : "=s"(s) : "v"(v) : "memory"); return s; }
__device__ __forceinline__ bool expired(unsigned t0, char* tmo, unsigned call) {
    const unsigned dt = now() - t0;
    if (dt < 200000u) return false;
    if (rfl(__hip_atomic_load((gu32*)tmo, __ATOMIC_RELAXED, __HIP_MEMORY_SCOPE_AGENT)) == call || dt > DEADLINE) {
        __hip_atomic_store((gu32*)tmo, call, __ATOMIC_RELAXED, __HIP_MEMORY_SCOPE_AGENT);
        return true;
    }
    return false;
}
#define PHYS 0x57413620u
__host__ __device__ __forceinline__ unsigned phys(unsigned l) { return (PHYS >> (4 * l)) & 7u; }
__host__ __device__ __forceinline__ int link_rank(int rank, int lk) { return lk < 3 ? (rank & 4) + ((rank + 1 + lk) & 3) : ((rank & 4) ^ 4) + ((rank + lk - 3) & 3); }
__device__ __forceinline__ int link_back(int lk) { return lk < 3 ? 2 - lk : 3 + ((3 - lk) & 3); }


__device__ __forceinline__ f32x4 bld(__amdgpu_buffer_rsrc_t rs, unsigned vo, unsigned so) { return __builtin_bit_cast(f32x4, __builtin_amdgcn_raw_buffer_load_b128(rs, vo, so, 0)); }
__device__ __forceinline__ void fetch_x(__amdgpu_buffer_rsrc_t rx, unsigned row, unsigned k, int lane, f32x4 (&r)[8]) {
    const unsigned vo = ((lane >> 2) * KDIM + 8 * ((lane & 3) ^ ((lane >> 5) << 1))) * 4u, so = (row * KDIM + k) * 4u;
#pragma unroll
    for (int st = 0; st < 4; ++st) { r[2 * st] = bld(rx, vo + (st & 1) * 128, so + (st >> 1) * 16 * KDIM * 4u); r[2 * st + 1] = bld(rx, vo + (st & 1) * 128 + 16, so + (st >> 1) * 16 * KDIM * 4u); }
}
__device__ __forceinline__ void pack_x(const f32x4 (&r)[8], u32x4 (&v)[4]) {
#pragma unroll
    for (int st = 0; st < 4; ++st) v[st] = cvt8(r[2 * st], r[2 * st + 1]);
}
__device__ __forceinline__ void fetch_w(__amdgpu_buffer_rsrc_t rw, unsigned col, unsigned k, int lane, f32x4 (&r)[8]) {
    const unsigned vo = (8 * (lane >> 3) * NLOC + 4 * (lane & 7)) * 4u, so = (k * NLOC + col) * 4u;
#pragma unroll
    for (int kk = 0; kk < 8; ++kk) r[kk] = bld(rw, vo, so + kk * NLOC * 4u);
}
__device__ __forceinline__ void pack_w(const f32x4 (&a)[8], int lane, LAS unsigned char* scr, u32x4 (&v)[4]) {
#pragma unroll
    for (int j = 0; j < 4; ++j) { u32x4 r; r.x = pk2(a[0][j], a[1][j]); r.y = pk2(a[2][j], a[3][j]); r.z = pk2(a[4][j], a[5][j]); r.w = pk2(a[6][j], a[7][j]);
        *(LAS u32x4*)(scr + (lane & 4) * 512 + (lane >> 3) * 256 + (lane & 3) * 64 + j * 16) = r; }
    asm volatile("s_waitcnt lgkmcnt(0)" ::: "memory");
#pragma unroll
    for (int st = 0; st < 4; ++st) v[st] = *(const LAS u32x4*)(scr + lane * 16 + st * 1024);
    asm volatile("s_waitcnt lgkmcnt(0)" ::: "memory");
}
__device__ __forceinline__ char* ready_base(const Params& P, int lane) {
    const int lk = min(lane, 6);
    return P.lw[lk] + OFF_RF + link_back(lk) * 2048 + PP * 60;
}

__device__ __forceinline__ void convert_role(const Params& P, unsigned callv, int t, int step, int nw, LAS unsigned char* scr) {
    const int lane = threadIdx.x & 63, rank = P.rank;
    char* const myws = P.me;
    const __amdgpu_buffer_rsrc_t rl = mk_rsrc(myws, WS_NEED);
    const __amdgpu_buffer_rsrc_t rx = mk_rsrc(P.x, 1024u * KDIM * 4u), rw = mk_rsrc(P.w, 1024u * KDIM * 4u);
    const __amdgpu_buffer_rsrc_t rn = mk_rsrc(TAB_NT, 4096u);
    char* const rfp = ready_base(P, lane);
    f32x4 r[8]; u32x4 v[4]; unsigned img, b, nn;
    auto fetch = [&](int tt) {
        const unsigned blk = tt >> 4, sub = tt & 15, kq = blk >> 4, code = (unsigned)(CVORD >> (4 * (blk & 15))) & 15u;
        img = code >> 1; b = (code & 1) * 32 + kq;
        const unsigned rc = (code & 1) * 128 + (sub & 3) * 32, k = (kq * 4 + (sub >> 2)) * 64;
        if (img < 4) fetch_x(rx, img * 256 + rc, k, lane, r); else fetch_w(rw, ((rank + img) & 3) * 256 + rc, k, lane, r);
        nn = __builtin_amdgcn_raw_buffer_load_b8(rn, lane < 7 ? lane : OOB, (img * 64 + b) * 8, 0);
    };
    fetch(t);
    const unsigned call = rfl(callv) + 1u, par = call & 1u;
    for (;;) {
        if (img < 4) pack_x(r, v); else pack_w(r, lane, scr, v);
        const unsigned slot = img < 4 ? rank * 4 + img : (rank + img) & 3, np = nn, in = (b >> 5) * HALFB + (b & 31u) * 65536u + (t & 15) * 4096u;
        const unsigned doff = (img < 4 ? OFF_X + (par * 32 + slot) * CHUNKB : OFF_W + (par * 8 + slot) * CHUNKB) + in;
        const unsigned foff = cnt_off(img < 4 ? slot : 32 + slot, b), sub = t & 15;
        SCHED;
        st16x4(v[0], v[1], v[2], v[3], rl, lane * 16, doff, 1024u);
        SCHED;
        t += step; step = nw;
        const bool more = t < 8192;
        if (more) { fetch(t); SCHED; WAIT_V(9); } else WAIT_V(0);
        if (lane < 7 && np < PP) st_sys(rfp + (np * 16 + sub) * 4 + par * RFP - PP * 60, call << 8);
        const unsigned o = add1(myws + foff, 1u);
        asm volatile("s_waitcnt vmcnt(0)" : "+v"(r[0]), "+v"(r[1]), "+v"(r[2]), "+v"(r[3]), "+v"(r[4]), "+v"(r[5]), "+v"(r[6]), "+v"(r[7]), "+v"(nn) :: "memory");
        if (got(o) + 1u == 16u * call && lane < 7 && np - PP < 255u - PP) st_sys(rfp + np * 4 + par * RFP, call << 8);
        if (!more) break;
    }
}


__device__ __forceinline__ void push_role(const Params& P, unsigned callv, int lk, int sub, LAS unsigned char* scr) {
    const int lane = threadIdx.x & 63, rank = P.rank;
    const __amdgpu_buffer_rsrc_t rx = mk_rsrc(P.x, 1024u * KDIM * 4u), rw = mk_rsrc(P.w, 1024u * KDIM * 4u);
    char* const pw = uni(P.lw[lk]);
    const __amdgpu_buffer_rsrc_t rp = mk_rsrc(pw, WS_NEED);
    f32x4 r[NPUSH + 1][8]; u32x4 v[4];
#pragma unroll
    for (int i = 0; i < NPUSH; ++i) {
        const unsigned rc = (i & 1) * 128 + (sub & 3) * 32, k = (i >> 2) * 256 + (sub >> 2) * 64;
        if (lk < 3 || (i & 2)) fetch_x(rx, (lk < 3 ? 1 + ((i >> 1) & 1) : 0) * 256 + rc, k, lane, r[i]); else fetch_w(rw, ((rank + lk + 1) & 3) * 256 + rc, k, lane, r[i]);
    }
    const unsigned par = (rfl(callv) + 1u) & 1u;
    unsigned pf = 0;
#pragma unroll
    for (int i = 0; i < NPUSH; ++i) {
        const bool isx = lk < 3 || (i & 2);
        if (isx) pack_x(r[i], v); else pack_w(r[i], lane, scr, v);
        const unsigned slot = isx ? rank * 4 + (lk < 3 ? 1 + ((i >> 1) & 1) : 0) : link_back(lk) + 1;
        st16x4(v[0], v[1], v[2], v[3], rp, lane * 16, (isx ? OFF_X + (par * 32 + slot) * CHUNKB : OFF_W + (par * 8 + slot) * CHUNKB) + (i & 1) * HALFB + (i >> 2) * 65536u + sub * 4096u, 1024u);
        const unsigned f = cnt_off(isx ? slot : 32 + slot, (i & 1) * 32 + (i >> 2));
        if (i) { WAIT_V(4); add0(pw + pf); }
        pf = f;
    }
    WAIT_V(0); add0(pw + pf);
}

__device__ __forceinline__ bool comm_role(const Params& P, unsigned call, int ci) {
    bool dead = false;
    const int lane = threadIdx.x & 63, wid = rfl(threadIdx.x >> 6);
    const int rank = P.rank, q = rank & 3, gb = rank & 4;
    const unsigned par = call & 1u, full = 16u * call, c24 = call & 0xffffffu;
    char* const myws = P.me;
    const int pi = wid * NCOMM + ci;
    if (wid == 7 && ci == 0) {
        const unsigned t0 = now();
        for (;;) {
            unsigned f;
            asm volatile("s_nop 4\n\tglobal_load_dword %0, %1, %2 sc0 sc1\n\ts_waitcnt vmcnt(0)" : "=&v"(f) : "v"(OFF_XF + min(lane, 6) * 4), "s"(myws) : "memory");
            const unsigned long long m = __ballot(ge(f, call));
            if (m == ~0ull) break;
            if (expired(t0, myws + OFF_TMO, call)) { dead = true; break; }
            if (m) __builtin_amdgcn_s_sleep(XSL); else __builtin_amdgcn_s_sleep(100);
        }
    } else if (pi / 7 < NPL) {
        const int lk = pi % 7, pk = pi / 7, src = link_rank(rank, lk), lb = link_back(lk);
        const unsigned nq = TAB_LEN[lk], ntask = (nq + (lk < 3 ? 0u : NYB)) * 16u;
        const __amdgpu_buffer_rsrc_t rl = mk_rsrc(myws, WS_NEED);
        char* const wss = uni(P.lw[lk]);
        const __amdgpu_buffer_rsrc_t rn = mk_rsrc(TAB_NT + 4096, sizeof TAB_NT - 4096u);
        char* const tkp = myws + OFF_TK + lk * 256;
        const unsigned tk0 = NPL - (call - 1u) * ntask;
        char* const dummy = myws + OFF_TK + 12 * 256 + pi * 64;
        char* const rfp = ready_base(P, lane);
        const unsigned vi = lane * 16, vy = (lane >> 5) * 4096 + (lane & 31) * 16;
        const unsigned qe0 = TAB_Q[lk * QS + lane], qe1 = TAB_Q[lk * QS + lane + 64], qe2 = TAB_Q[lk * QS + lane + 128];
        unsigned tn = pk, win = 0, wbase = 0;
        bool hc = false;
        unsigned cdf = 0, cni = 127;
        const unsigned t0 = now();
        unsigned rdy, fi, wa, sof, dof, df, ni;
        auto look = [&]() {
            const unsigned pos = tn >> 4, sub = tn & 15;
            fi = pos < PP ? tn : (pos < nq ? pos : (tn >> 3) - nq) + PP * 15;
            const unsigned wi = fi - wbase, rw = wi < 64u ? rdl(win, wi) : 0u;
            rdy = tn < ntask && ((rw >> 8) == c24 || dead);
            wa = lane < 32 || pos < PP ? OFF_RF + par * RFP + (lk * 512 + fi + lane) * 4 : OOB;
            df = 0; ni = 127;
            if (pos < nq) {
                const unsigned e0 = rdl(qe0, pos & 63u), e1 = rdl(qe1, pos & 63u), e2 = rdl(qe2, pos & 63u), e = pos < 64u ? e0 : (pos < 128u ? e1 : e2);
                const bool vs = (e & 0x7e0u) == 0xe0u;
                const unsigned b = e & 63u, who = (e >> 8) & 7u, in = (b >> 5) * HALFB + (b & 31u) * 65536u + (vs ? (tn & 6u) * 2u + (sub >> 3) * 2u + (tn & 1u) : sub) * 4096u;
                ni = e >> 11;
                if (who == 4u) { sof = OFF_W + (par * 8 + q) * CHUNKB + in; dof = OFF_W + (par * 8 + lk + 1) * CHUNKB + in; df = cnt_off(33 + lk, b); }
                else { const unsigned G = (who ? gb + ((q + who) & 3) : src) * 4 + ((e >> 6) & 3u); sof = dof = OFF_X + (par * 32 + G) * CHUNKB + in; df = cnt_off(vs && (tn & 8u) ? 40u + lk : G, b) + vs; }
            } else {
                const unsigned id = rw & 127u;
                df = rw & 128u;
                sof = OFF_Y + ((par * 4 + lb - 3) * 128 + id) * 32768u + (sub & 7u) * 4096u;
                dof = (phys(((id & 24u) ? gb ^ 4 : gb) + (id >> 5)) * 1024u + ((id >> 2) & 7u) * 128u + (id & 1u) * 64u + (sub & 7u) * 8u) * 4096u + (src & 3) * 1024u + (id & 2u) * 256u;
            }
            rdy = rfl(rdy); fi = rfl(fi); sof = rfl(sof); dof = rfl(dof); df = rfl(df); ni = rfl(ni);
        };
        look();
        for (;;) {
            const bool ready = rdy;
            if (!ready && !hc) {
                if (tn >= ntask) break;
                for (unsigned n = 1;; ++n) {
                    asm volatile("s_nop 4\n\tbuffer_load_dword %0, %1, %2, 0 offen sc0 sc1\n\ts_waitcnt vmcnt(0)" : "=&v"(win) : "v"(wa), "s"(rl) : "memory");
                    if ((rfl(win) >> 8) == c24) break;
                    if ((n & 63u) == 0u && (dead = expired(t0, myws + OFF_TMO, call))) break;
                    __builtin_amdgcn_s_sleep(ISLEEP);
                }
                wbase = fi;
                look();
                continue;
            }
            u32x4 w0, w1, w2, w3;
            unsigned tv = add1(ready ? tkp : dummy, 1u);
            {
                const __amdgpu_buffer_rsrc_t rr = mk_rsrc(wss, ready ? WS_NEED : 0u);
                asm volatile("s_nop 4\n\tbuffer_load_dword %4, %8, %9, 0 offen sc0 sc1\n\t"
                             "buffer_load_dwordx4 %0, %5, %6, %7 offen sc0 sc1\n\tbuffer_load_dwordx4 %1, %5, %6, %7 offen offset:1024 sc0 sc1\n\t"
                             "buffer_load_dwordx4 %2, %5, %6, %7 offen offset:2048 sc0 sc1\n\tbuffer_load_dwordx4 %3, %5, %6, %7 offen offset:3072 sc0 sc1\n\t"
                             "s_waitcnt vmcnt(6)"
                             : "=&v"(w0), "=&v"(w1), "=&v"(w2), "=&v"(w3), "=&v"(win) : "v"(vi), "s"(rr), "s"(sof), "v"(wa), "s"(rl) : "memory");
            }
            wbase = fi;
            const bool hop = hc && cni != 127u, cy = (tn >> 4) >= nq, xf = ready && tn == ntask - XEARLY;
            const unsigned hni = cni;
            unsigned o = 0;
            if (hop) o = add1(myws + cdf, 1u); else add0(hc ? myws + (cdf & ~1u) : dummy, 1u + (cdf & 1u));
            const __amdgpu_buffer_rsrc_t rs = mk_rsrc(cy ? (const void*)P.out : (const void*)myws, !ready ? 0u : (cy ? 8192u * NLOC * 4u : WS_NEED));
            const unsigned pitch = cy ? 8192u : 1024u, vo = cy ? vy : vi, so = dof;
            hc = ready && !cy; cdf = df; cni = ni;
            const unsigned esof = sof, efi = fi;
            asm volatile("s_waitcnt vmcnt(5)" : "+v"(win), "+v"(tv) :: "memory");
            if (ready) tn = got(tv) + tk0;
            look();
            asm volatile("s_waitcnt vmcnt(1)" : "+v"(w0), "+v"(w1), "+v"(w2), "+v"(w3), "+s"(rdy), "+s"(fi), "+s"(sof), "+s"(dof), "+s"(df), "+s"(ni), "+v"(wa) :: "memory");
            if (cy && cdf && ready) {
                const u32x4 ma = __builtin_elementwise_max(__builtin_elementwise_max(w0, w1), __builtin_elementwise_max(w2, w3));
                if (__ballot(max(max(ma.x, ma.y), max(ma.z, ma.w)) == ~0u)) {
                    while ((rfl(ld_sys(myws + OFF_RF + par * RFP + (lk * 512 + efi + 128) * 4)) >> 8) != c24 && !(dead = expired(t0, myws + OFF_TMO, call))) __builtin_amdgcn_s_sleep(1);
                    const __amdgpu_buffer_rsrc_t rr = mk_rsrc(wss, WS_NEED);
                    asm volatile("s_nop 4\n\tbuffer_load_dwordx4 %0, %4, %5, %6 offen sc0 sc1\n\tbuffer_load_dwordx4 %1, %4, %5, %6 offen offset:1024 sc0 sc1\n\t"
                                 "buffer_load_dwordx4 %2, %4, %5, %6 offen offset:2048 sc0 sc1\n\tbuffer_load_dwordx4 %3, %4, %5, %6 offen offset:3072 sc0 sc1\n\ts_waitcnt vmcnt(0)"
                                 : "=&v"(w0), "=&v"(w1), "=&v"(w2), "=&v"(w3) : "v"(vi), "s"(rr), "s"(esof) : "memory");
                }
            }
            st16x4(w0, w1, w2, w3, rs, vo, so, pitch);
            if (hop) {
                WAIT_V(4);
                if (got(o) + 1u == full) {
                    const unsigned np = __builtin_amdgcn_raw_buffer_load_b8(rn, lane < 7 ? hni * 8 + lane : OOB, 0, 0);
                    if (lane < 7 && np != 255u) st_sys(rfp + np * 4 + par * RFP, call << 8);
                }
            }
            if (xf && lane == 0) st_sys(wss + OFF_XF + lb * 4, call);
        }
    }
    return dead;
}

__device__ __forceinline__ unsigned ld_atom(const char* p) {
    unsigned v;
    asm volatile("global_atomic_add %0, %1, %2, off sc0 sc1\n\ts_waitcnt vmcnt(0)" : "=&v"(v) : "v"(p), "v"(0u) : "memory");
    return v;
}
__device__ __forceinline__ int poll_avail(const char* ws, unsigned fa, unsigned fb, int s0, unsigned call, int lane, bool& bad, bool hurry, unsigned& kn, int lazy = 0) {
    const int base = s0 & 16;
    const char* p = ws + ((lane & 32) ? fb : fa) + ((lane & 16) ? FSTR : 0u) + (base + (lane & 15)) * 4;
    const unsigned t0 = now();
    int n;
    for (;;) {
        bool ok = (lane & 15) < s0 - base;
        if ((int)((kn >> ((lane >> 4) * 8)) & 255u) <= s0) ok |= ge(ld_atom(p), 16u * call);
        const unsigned long long m = ~__ballot(ok);
        n = NST;
#pragma unroll
        for (int g = 0; g < 4; ++g) {
            const int k = max((int)((kn >> (8 * g)) & 255u), base + __builtin_ctz((unsigned)(m >> (16 * g)) | 0x10000u));
            kn = (kn & ~(255u << (8 * g))) | (unsigned)k << (8 * g);
            n = min(n, k);
        }
        if (n > s0) break;
        if (expired(t0, (char*)ws + OFF_TMO, call)) { n = NST; bad = true; break; }
        if (lazy) __builtin_amdgcn_s_sleep(OLAZY);
        if (hurry) __builtin_amdgcn_s_sleep(PSLEEP); else __builtin_amdgcn_s_sleep(PSLOW);
    }
    return n;
}

__device__ __forceinline__ void gemm_role(const Params& P, unsigned call, unsigned job, LAS unsigned char* lds, unsigned si, bool& bad) {
    const int tid = threadIdx.x, wid = rfl(tid >> 6), lane = tid & 63, wr = wid >> 2, wc = wid & 3, fr = lane & 15, fq = lane >> 4;
    const int rank = P.rank, q = rank & 3, gb = rank & 4;
    const int ct = job & 7, mr = (job >> 5) & 7, ah = (job >> 8) & 1, bh = (job >> 9) & 1, rq = (job >> 12) & 1; const bool quarter = (job >> 11) & 1;
    const int G = ((gb ^ (mr & 4)) + ((q + mr) & 3)) * 4 + ((job >> 3) & 3);
    const unsigned par = call & 1u;
    char* const myws = P.me;
    const char* const cA = myws + OFF_X + (size_t)(par * 32 + G) * CHUNKB + ah * HALFB;
    const char* const cB = myws + OFF_W + (size_t)(par * 8 + ct) * CHUNKB + bh * HALFB;
    const unsigned fA = cnt_off(rq ? 39 + mr : G, ah * 32), fB = cnt_off(32 + ct, bh * 32);
    unsigned kn = quarter ? 0x20002000u : 0u;
    LAS volatile int* const availw = (LAS volatile int*)(lds + STAGE_BYTES);
    const unsigned voff = (unsigned)tid * 16u;
    const unsigned ldsw = (unsigned)wid * 1024u;
    const int aoff = lds_byte(wr * 64 + fr, fq * 8), boff = wc * 4096 + lane * 16;
    constexpr size_t kstep = HTB, hstep = HALFB;
#define SA(b, h) (((b) * 2 + (h)) * HTB)
#define SB(b, h) ((4 + (b) * 2 + (h)) * HTB)
#define STAGE(bufoff, gbase) do { _Pragma("unroll") for (int _i = 0; _i < 2; ++_i) \
        __builtin_amdgcn_global_load_lds((const unsigned*)((const char*)(gbase) + voff + _i * 8192), (LAS unsigned*)(lds + (bufoff) + ldsw + _i * 8192), 16, 0, 0); } while (0)
#define LDA(dst, b, h) do { _Pragma("unroll") for (int m = 0; m < 4; ++m) _Pragma("unroll") for (int k = 0; k < 2; ++k) dst[m][k] = *(const LAS f16x8*)(lds + SA(b, h) + aoff + m * 2048 + k * 1024); } while (0)
#define LDB(dst, b, h) do { _Pragma("unroll") for (int n = 0; n < 2; ++n) _Pragma("unroll") for (int k = 0; k < 2; ++k) dst[n][k] = *(const LAS f16x8*)(lds + SB(b, h) + boff + n * 2048 + k * 1024); } while (0)
#define MM(ac, At, Bt) do { _Pragma("unroll") for (int k = 0; k < 2; ++k) _Pragma("unroll") for (int m = 0; m < 4; ++m) _Pragma("unroll") for (int n = 0; n < 2; ++n) \
        ac[m][n] = __builtin_amdgcn_mfma_f32_16x16x32_f16(Bt[n][k], At[m][k], ac[m][n], 0, 0, 0); } while (0)
#define MMA(ai, bj, At, Bt) do { __builtin_amdgcn_s_setprio(1); MM(acc[ai][bj], At, Bt); __builtin_amdgcn_s_setprio(0); } while (0)
#define NEED(s) do { if ((s) >= avail) { \
            if (wid == 0) { const int _a = poll_avail(myws, fA, fB, (s), call, lane, bad, (s) >= (si ? PFROM : EFROM), kn); if (lane == 0) *availw = _a; WAIT_L(0); } \
            asm volatile("" ::: "memory"); BAR; asm volatile("" ::: "memory"); \
            avail = rfl(*availw); } } while (0)
    int avail = 0;
    const bool remote = ct >= 4;
    const __amdgpu_buffer_rsrc_t ro = mk_rsrc(remote ? (const void*)(myws + OFF_Y) : (const void*)P.out, 8192u * NLOC * 4u);
    const unsigned yid = (G & 15) * 8 + ah * 4 + bh * 2 + rq;
    const unsigned obase = remote ? ((par * 4 + ct - 4) * 128 + yid) * 32768u : ((phys(G >> 2) * 1024u + (G & 3) * 256 + ah * 128 + rq * 64) * NLOC + ct * 256 + bh * 128) * 4u;
    char* const ytk = myws + OFF_TK + (4 + ct) * 256;
    char* const yrf = P.lw[remote ? ct - 1 : 0] + OFF_RF + par * RFP + (link_back(remote ? ct - 1 : 0) * 512 + TAB_LEN[link_back(remote ? ct - 1 : 0)]) * 4 + PP * 60;
    if (!quarter) {
        f32x4 acc[2][2][4][2];
#pragma unroll
        for (int a = 0; a < 2; ++a)
#pragma unroll
            for (int b = 0; b < 2; ++b)
#pragma unroll
                for (int m = 0; m < 4; ++m)
#pragma unroll
                    for (int n = 0; n < 2; ++n) acc[a][b][m][n] = (f32x4){0.f, 0.f, 0.f, 0.f};
        f16x8 At[4][2], B0[2][2], B1[2][2];
        NEED(0);
#define WV_A WAIT_V(10)
        STAGE(SB(0, 0), cB); STAGE(SA(0, 0), cA); STAGE(SB(0, 1), cB + hstep); STAGE(SA(0, 1), cA + hstep);
        STAGE(SB(1, 0), cB + kstep); STAGE(SA(1, 0), cA + kstep); STAGE(SB(1, 1), cB + hstep + kstep);
        if (wr == 1) BAR;
        WAIT_V(10); BAR;
        BAR;
        LDB(B0, 0, 0);
        for (int t = 0; t < NKT; t += 2) {
            const bool last = (t == NKT - 2);
            const char* a1 = cA + (size_t)(t + 1) * kstep;
            const char* a2 = cA + (size_t)(last ? t : t + 2) * kstep; const char* b2 = cB + (size_t)(last ? t : t + 2) * kstep;
            const char* a3 = a2 + kstep; const char* b3 = b2 + kstep;
            if (!last) NEED((t + 2) >> 2);
            LDA(At, 0, 0); STAGE(SA(1, 1), a1 + hstep);
            WV_A; WAIT_L(8); BAR; WAIT_L(0); MMA(0, 0, At, B0); BAR; SCHED;
            LDB(B1, 0, 1); STAGE(SB(0, 0), b2);
            WV_A; BAR; WAIT_L(0); MMA(0, 1, At, B1); BAR;
            LDA(At, 0, 1); STAGE(SA(0, 0), a2);
            WV_A; BAR; WAIT_L(0); MMA(1, 0, At, B0); BAR; SCHED;
            LDB(B0, 1, 0); STAGE(SB(0, 1), b2 + hstep);
            WV_A; BAR; MMA(1, 1, At, B1); BAR; SCHED;
            LDA(At, 1, 0); STAGE(SA(0, 1), a2 + hstep);
            WV_A; WAIT_L(8); BAR; WAIT_L(0); MMA(0, 0, At, B0); BAR; SCHED;
            LDB(B1, 1, 1); STAGE(SB(1, 0), b3);
            WV_A; BAR; WAIT_L(0); MMA(0, 1, At, B1); BAR;
            LDA(At, 1, 1); STAGE(SA(1, 0), a3);
            WV_A; BAR; WAIT_L(0); MMA(1, 0, At, B0); BAR; SCHED;
            LDB(B0, 0, 0); STAGE(SB(1, 1), b3 + hstep);
            WV_A; BAR; MMA(1, 1, At, B1); BAR; SCHED;
        }
        WAIT_V(0);
        if (wr == 0) BAR;
        BAR;
        int tz = threadIdx.x; asm volatile("" : "+v"(tz));
        const int ln = tz & 63, efr = ln & 15, efq = ln >> 4;
        const unsigned ovo = remote ? (ln >> 5) * 65536u + (ln & 31) * 16u : ln * 16u, opitch = remote ? 512u : NLOC * 4u;
#pragma unroll
        for (int ai = 0; ai < 2; ++ai) {
#pragma unroll
            for (int m = 0; m < 4; ++m)
#pragma unroll
                for (int bj = 0; bj < 2; ++bj)
#pragma unroll
                    for (int n = 0; n < 2; ++n) {
                        const int row = wr * 64 + m * 16 + efr, ch = bj * 32 + wc * 8 + n * 4 + efq;
                        *(LAS f32x4*)(lds + row * 1024 + ((ch ^ (row & 7)) << 4)) = acc[ai][bj][m][n];
                    }
            unsigned slot = 0;
            if (remote && wid == 0) slot = add1(ytk, 4u);
            WAIT_L(0); BAR; asm volatile("" ::: "memory");
#pragma unroll
            for (int i0 = 0; i0 < 16; i0 += 8) {
                u32x4 o[8];
#pragma unroll
                for (int i = 0; i < 8; ++i) o[i] = *(const LAS u32x4*)(lds + (wid * 16 + i0 + i) * 1024 + ((ln ^ i) << 4));
                const unsigned so = obase + (remote ? ai * 131072u : ai * 128u * opitch) + (unsigned)(wid * 16 + i0) * opitch;
                st16x4(o[0], o[1], o[2], o[3], ro, ovo, so, opitch);
                st16x4(o[4], o[5], o[6], o[7], ro, ovo, so + 4 * opitch, opitch);
            }
            if (remote) WAIT_V(0);
            WAIT_L(0); BAR; asm volatile("" ::: "memory");
            if (remote && wid == 0) {
                const unsigned sl = got(slot) - (call - 1u) * (2u * NYB);
                if (ln < 4) st_sys(yrf + min(sl + ln, 127u) * 4, (call << 8) | (yid + ai * 4 + ln));
            }
        }
    } else if (job & 0x400u) {
        if (wid == 0) {
            if (lane < 12) availw[lane] = 0;
            kn = 0x20002020u;
            for (int a = 0; a < NST;) a = poll_avail(myws, fA, fB, a, call, lane, bad, false, kn);
            WAIT_L(0);
        }
        if (remote) { const u32x4 mk = {~0u, ~0u, ~0u, ~0u}; st16x4(mk, mk, mk, mk, ro, lane * 16, obase + wid * 4096u, 1024u); }
        unsigned slot = 0;
        asm volatile("" ::: "memory"); BAR; asm volatile("" ::: "memory");
        const unsigned t0 = now();
        unsigned spins = 0;
#define HSPIN(sl) { if ((++spins & 4095u) == 0u && now() - t0 > 2 * DEADLINE) { bad = true; break; } __builtin_amdgcn_s_sleep(sl); }
        f32x4 acc[4][2];
#pragma unroll
        for (int m = 0; m < 4; ++m)
#pragma unroll
            for (int n = 0; n < 2; ++n) acc[m][n] = (f32x4){0.f, 0.f, 0.f, 0.f};
        if (wr) {
            const char* ga = cA + rq * 8192 + wc * 65536 + lane * 16;
            for (int s = wc; s < NST; s += 4, ga += 4 * 65536) {
                int av;
                while ((av = rfl(*availw)) < s) HSPIN(LSLEEP);
                if (av == s) { kn = 0x20202000u; if (!remote) __builtin_amdgcn_s_sleep(OLAZY); av = poll_avail(myws, fA, fB, s, call, lane, bad, s >= HFROM, kn, remote ? 0 : OLAZY); if (lane == 0) *availw = av; }
                while (__ballot(availw[8 + (lane & 3)] < s - 3)) HSPIN(LSLEEP);
                if (s == NST - 1 && remote) slot = add1(ytk, 1u);
                asm volatile("" ::: "memory");
#pragma unroll
                for (int i = 0; i < XNA; ++i) {
                    const unsigned* g = (const unsigned*)(ga + (i >> 1) * 16384 + (i & 1) * 4096); LAS unsigned* l = (LAS unsigned*)(lds + wc * 32768 + i * 4096);
                    __builtin_amdgcn_global_load_lds(g, l, 16, 0, 0); __builtin_amdgcn_global_load_lds(g, l, 16, 1024, 0);
                    __builtin_amdgcn_global_load_lds(g, l, 16, 2048, 0); __builtin_amdgcn_global_load_lds(g, l, 16, 3072, 0);
                }
                const unsigned fw = STAGE_BYTES + 16 + wc * 4;
                asm volatile("s_mov_b64 exec, 1\n\ts_waitcnt vmcnt(24)\n\tds_write_b32 %0, %1\n\ts_waitcnt vmcnt(16)\n\tds_write_b32 %0, %2\n\t"
                             "s_waitcnt vmcnt(8)\n\tds_write_b32 %0, %3\n\ts_waitcnt vmcnt(0)\n\tds_write_b32 %0, %4\n\ts_mov_b64 exec, -1\n\ts_waitcnt lgkmcnt(0)"
                             :: "v"((unsigned)(size_t)(lds + fw)), "v"(4 * s + 1), "v"(4 * s + 2), "v"(4 * s + 3), "v"(4 * s + 4) : "memory");
            }
            if (remote && wid == 7) {
                slot = min(got(slot) - (call - 1u) * (2u * NYB), 127u);
                __builtin_amdgcn_s_sleep(EDELAY);
                if (lane == 0) st_sys(yrf + slot * 4, (call << 8) | 128u | yid);
            }
        } else {
            f16x8 Bq[8][2][2], At[2][2][4];
#define HB4(c, u) c(Bq[u][0][0]), c(Bq[u][0][1]), c(Bq[u][1][0]), c(Bq[u][1][1])
#define HLD1(c, u, n, k, kt) asm volatile("buffer_load_dwordx4 %0, %1, %2, %3 offen offset:%4" : c(Bq[u][n][k]) : "v"(boff), "s"(rb), "s"((kt) * HTB), "n"(n * 2048 + k * 1024) : "memory")
            const __amdgpu_buffer_rsrc_t rb = mk_rsrc(cB, HALFB);
            asm volatile("s_nop 4" :: "s"(rb));
#pragma unroll
            for (int u = 0; u < 8; ++u) { HLD1("=v", u, 0, 0, u); HLD1("=v", u, 1, 0, u); HLD1("=v", u, 0, 1, u); HLD1("=v", u, 1, 1, u); }
            const unsigned l0 = (unsigned)(size_t)lds, lw = l0 + STAGE_BYTES;
#define HA4(c, b, k) c(At[b][k][0]), c(At[b][k][1]), c(At[b][k][2]), c(At[b][k][3])
#define HA8(c, b) HA4(c, b, 0), HA4(c, b, 1)
#define HRD "ds_read_b128 %0, %[a] offset:%[o]\n\tds_read_b128 %1, %[a] offset:%[o]+2048\n\tds_read_b128 %2, %[a] offset:%[o]+4096\n\tds_read_b128 %3, %[a] offset:%[o]+6144\n\t" \
            "ds_read_b128 %4, %[a] offset:%[o]+1024\n\tds_read_b128 %5, %[a] offset:%[o]+3072\n\tds_read_b128 %6, %[a] offset:%[o]+5120\n\tds_read_b128 %7, %[a] offset:%[o]+7168\n\t"
#define HM(u, n, k, ld) do { SCHED; _Pragma("unroll") for (int m = 0; m < 4; ++m) \
            acc[m][n] = __builtin_amdgcn_mfma_f32_16x16x32_f16(Bq[u][n][k], At[u & 1][k][m], acc[m][n], 0, 0, 0); SCHED; if (ld) HLD1("+v", u, n, k, kt + 8); } while (0)
            while (rfl(availw[4]) <= 0) HSPIN(HSLEEP);
            asm volatile(HRD : HA8("=&v", 0) : [a] "v"(l0 + aoff), [o] "n"(0) : "memory");
#define HTILES(it, last) _Pragma("unroll") for (int u = 0; u < 8; ++u) { \
                const unsigned an = l0 + aoff + (((it) + (u == 7)) & 1) * 65536; \
                const int kt = (it) * 8 + u, fs = 4 + (((kt + 1) >> 2) & 3); \
                int nf; \
                if ((u & 3) == 3 && !last) asm volatile("s_mov_b64 exec, 1\n\tds_write_b32 %0, %1\n\ts_mov_b64 exec, -1" :: "v"(lw + 32 + wc * 4), "v"((kt >> 2) + 1) : "memory"); \
                if (last && u == 7) asm volatile("s_waitcnt vmcnt(0) lgkmcnt(4)" : HA4("+v", 1, 0), HB4("+v", u) :: "memory"); \
                else asm volatile("ds_read_b32 %[f], %[w]\n\t" HRD "s_waitcnt vmcnt(%[n]) lgkmcnt(13)" : HA8("=&v", ~u & 1), HA4("+v", u & 1, 0), HB4("+v", u), [f] "=&v"(nf) \
                        : [a] "v"(an), [o] "n"(((u + 1) & 7) * 8192), [w] "v"(lw + fs * 4), [n] "n"(last ? 28 - 4 * u : 28) : "memory"); \
                HM(u, 0, 0, !last); HM(u, 1, 0, !last); \
                asm volatile("s_waitcnt lgkmcnt(%4)" : HA4("+v", u & 1, 1) : "n"(last && u == 7 ? 0 : 9) : "memory"); \
                HM(u, 0, 1, !last); HM(u, 1, 1, !last); \
                if (last && u == 7) break; \
                asm volatile("s_waitcnt lgkmcnt(8)" : "+v"(nf) :: "memory"); \
                if (rfl(nf) <= kt + 1) { \
                    while (rfl(availw[fs]) <= kt + 1) HSPIN(HSLEEP); \
                    asm volatile(HRD : HA8("+v", ~u & 1) : [a] "v"(an), [o] "n"(((u + 1) & 7) * 8192) : "memory"); \
                } }
            for (int it = 0; it < 15; ++it) { HTILES(it, false) }
            HTILES(15, true)
        }
        WAIT_L(0); BAR; asm volatile("" ::: "memory");
        int tz = threadIdx.x; asm volatile("" : "+v"(tz));
        const int ln = tz & 63, efr = ln & 15, efq = ln >> 4;
        const unsigned opitch = remote ? 512u : NLOC * 4u, ovo = (ln >> 5) * opitch + (ln & 31) * 16u;
        if (!wr) {
#pragma unroll
            for (int m = 0; m < 4; ++m)
#pragma unroll
                for (int n = 0; n < 2; ++n) {
                    const int row = m * 16 + efr, ch = wc * 8 + n * 4 + efq;
                    *(LAS f32x4*)(lds + row * 512 + ((ch ^ (row & 7)) << 4)) = acc[m][n];
                }
        }
        WAIT_L(0); BAR; asm volatile("" ::: "memory");
        {
            u32x4 o[4];
#pragma unroll
            for (int i = 0; i < 4; ++i) { const int row = wid * 8 + 2 * i + (ln >> 5); o[i] = *(const LAS u32x4*)(lds + row * 512 + (((ln & 31) ^ (row & 7)) << 4)); }
            st16x4(o[0], o[1], o[2], o[3], ro, ovo, obase + (unsigned)(wid * 8) * opitch, 2 * opitch);
        }
        if (remote) {
            WAIT_V(0); BAR; asm volatile("" ::: "memory");
            if (wid == 7 && ln == 0) st_sys(yrf + slot * 4 + 512, call << 8);
        }
    } else {
        f32x4 acc[4][2];
#pragma unroll
        for (int m = 0; m < 4; ++m)
#pragma unroll
            for (int n = 0; n < 2; ++n) acc[m][n] = (f32x4){0.f, 0.f, 0.f, 0.f};
        f16x8 A0[4][2], A1[4][2], B0[2][2], B1[2][2];
#define QSTAGE(s, kt) do { STAGE(SA(0, s), cA + (size_t)(kt) * kstep); STAGE(SB(0, s), cB + (size_t)(kt) * kstep); } while (0)
#define QM(k, m0, Ac, Bc) do { _Pragma("unroll") for (int m = m0; m < m0 + 2; ++m) _Pragma("unroll") for (int n = 0; n < 2; ++n) \
            acc[m][n] = __builtin_amdgcn_mfma_f32_16x16x32_f16(Bc[n][k], Ac[m][k], acc[m][n], 0, 0, 0); } while (0)
#define QG(mask) do { _Pragma("unroll") for (int _j = 0; _j < 4; ++_j) { __builtin_amdgcn_sched_group_barrier(0x008, 1, 0); __builtin_amdgcn_sched_group_barrier(mask, 1, 0); } SCHED; } while (0)
#define QG2 do { _Pragma("unroll") for (int _j = 0; _j < 6; ++_j) { __builtin_amdgcn_sched_group_barrier(0x008, 1, 0); __builtin_amdgcn_sched_group_barrier(0x100, 2, 0); } __builtin_amdgcn_sched_group_barrier(0x008, 6, 0); SCHED; } while (0)
#define QG3 do { _Pragma("unroll") for (int _j = 0; _j < 6; ++_j) { __builtin_amdgcn_sched_group_barrier(0x008, 1, 0); __builtin_amdgcn_sched_group_barrier(0x100, 2, 0); } __builtin_amdgcn_sched_group_barrier(0x008, 10, 0); SCHED; } while (0)
#define QM16(Ac, Bc) do { QM(0, 0, Ac, Bc); QM(0, 2, Ac, Bc); QM(1, 0, Ac, Bc); QM(1, 2, Ac, Bc); } while (0)
#define QSTEP(s, kt, Ac, Bc, An, Bn) do { WAIT_V(8); WAIT_L(0); BAR; SCHED; \
            QSTAGE(s, kt); QM(0, 0, Ac, Bc); QG(0x020); \
            LDB(Bn, 0, ((s) + 1) & 3); LDA(An, 0, ((s) + 1) & 3); QM(0, 2, Ac, Bc); QM(1, 0, Ac, Bc); QM(1, 2, Ac, Bc); QG2; } while (0)
#define QDRAIN(wv, s, Ac, Bc, An, Bn) do { WAIT_V(wv); WAIT_L(0); BAR; SCHED; LDB(Bn, 0, (s) + 1); LDA(An, 0, (s) + 1); QM16(Ac, Bc); QG3; } while (0)
        NEED(0);
        QSTAGE(0, 0); QSTAGE(1, 1); QSTAGE(2, 2); QSTAGE(3, 3);
        WAIT_V(12); BAR; SCHED;
        LDB(B0, 0, 0); LDA(A0, 0, 0);
        int s = 0;
        for (; s < NST - 1; ++s) {
            NEED(s + 1);
            QSTEP(0, 4 * s + 4, A0, B0, A1, B1);
            QSTEP(1, 4 * s + 5, A1, B1, A0, B0);
            QSTEP(2, 4 * s + 6, A0, B0, A1, B1);
            QSTEP(3, 4 * s + 7, A1, B1, A0, B0);
        }
        QDRAIN(8, 0, A0, B0, A1, B1);
        QDRAIN(4, 1, A1, B1, A0, B0);
        QDRAIN(0, 2, A0, B0, A1, B1);
        WAIT_L(0); BAR; SCHED;
        QM16(A1, B1);
        int tz = threadIdx.x; asm volatile("" : "+v"(tz));
        const int ln = tz & 63, efr = ln & 15, efq = ln >> 4;
        const unsigned opitch = remote ? 512u : NLOC * 4u, ovo = (ln >> 5) * opitch + (ln & 31) * 16u;
#pragma unroll
        for (int m = 0; m < 4; ++m)
#pragma unroll
            for (int n = 0; n < 2; ++n) {
                const int row = wr * 64 + m * 16 + efr, ch = wc * 8 + n * 4 + efq;
                *(LAS f32x4*)(lds + row * 512 + ((ch ^ (row & 7)) << 4)) = acc[m][n];
            }
        unsigned slot = 0;
        if (remote && wid == 0) slot = add1(ytk, 2u);
        WAIT_L(0); BAR; asm volatile("" ::: "memory");
        {
            u32x4 o[8];
#pragma unroll
            for (int i = 0; i < 8; ++i) { const int row = wid * 16 + 2 * i + (ln >> 5); o[i] = *(const LAS u32x4*)(lds + row * 512 + (((ln & 31) ^ (row & 7)) << 4)); }
            const unsigned so = obase + (unsigned)(wid * 16) * opitch;
            st16x4(o[0], o[1], o[2], o[3], ro, ovo, so, 2 * opitch);
            st16x4(o[4], o[5], o[6], o[7], ro, ovo, so + 8 * opitch, 2 * opitch);
        }
        if (remote) {
            WAIT_V(0); BAR; asm volatile("" ::: "memory");
            if (wid == 0) { const unsigned sl = got(slot) - (call - 1u) * (2u * NYB); if (ln < 2) st_sys(yrf + min(sl + ln, 127u) * 4, (call << 8) | (yid + ln)); }
        }
    }
}

__global__ __launch_bounds__(512, 2) void ag_gemm_kernel(const Params P) {
    extern __shared__ __attribute__((aligned(16))) unsigned char shm[];
    char* const myws = P.me;
    const unsigned wid = rfl(threadIdx.x >> 6), cv = blockIdx.x - NCOMM, cw = OFF_CALL + (blockIdx.x * 8 + wid) * 4;
    const unsigned callv = __builtin_amdgcn_raw_buffer_load_b32(mk_rsrc(myws, 65536u), 0, cw, AUX_WT);
    if (cv < 128u && wid - 1u < (unsigned)(CVW * (1 + XR))) {
        convert_role(P, callv, (wid - 1) * 128 + cv, wid > CVW ? 8192 : 128 * CVW * (1 + XR), 128 * CVW, (LAS unsigned char*)shm + wid * 4096);
    }
    if (NPUSH && blockIdx.x < NCOMM && wid - 3u < 2u) {
        const unsigned pi = (wid - 3) * NCOMM + blockIdx.x;
        push_role(P, callv, pi % 7, pi / 7, (LAS unsigned char*)shm + wid * 4096);
    }
    unsigned bi = blockIdx.x; asm volatile("" : "+s"(bi));
    const unsigned job = TAB_JOB[bi], kind = (job >> 10) & 3u;
    const unsigned call = rfl(callv) + 1u;
    if ((threadIdx.x & 63) == 0) st_sys(myws + cw, call);
    bool bad = false;
    if (kind == 0u) bad = comm_role(P, call, job & 255u);
    else {
        if (wid == 0) {
            __builtin_amdgcn_fence(__ATOMIC_ACQUIRE, "");
            asm volatile("s_waitcnt vmcnt(0)" ::: "memory");
        }
        for (unsigned jb = job, si = 0; jb & 0xc00u; jb >>= 16, si += 4) {
            gemm_role(P, call, jb, (LAS unsigned char*)shm, si, bad);
        }
    }
    if (bad) {
        __hip_atomic_store((gu32*)(myws + OFF_TMO), call, __ATOMIC_RELAXED, __HIP_MEMORY_SCOPE_AGENT);
        const unsigned t0 = now();
        while (now() - t0 < 5000000u) __builtin_amdgcn_s_sleep(64);
        st_sys((char*)P.out, 0x7fc00000u);
    }
}
}

extern "C" void kernel_launch(int rank, int nranks, ncclComm_t comm,
                              hipStream_t stream,
                              void* const* d_in, const int* in_sizes, int n_in,
                              void* d_out, int out_size,
                              const int* in_dtypes, int out_dtype,
                              void* const* d_out_all,
                              void* const* d_ws_all, size_t ws_bytes) {
    if (nranks != NR || ws_bytes < WS_NEED || (unsigned)rank >= (unsigned)NR) { fprintf(stderr, "unsupported configuration\n"); return; }
    static bool attr_set[NR] = {};
    if (!attr_set[rank]) {
        HIP_CHECK(hipFuncSetAttribute((const void*)ag_gemm_kernel, hipFuncAttributeMaxDynamicSharedMemorySize, LDS_BYTES));
        attr_set[rank] = true;
    }
    Params P;
    memset(&P, 0, sizeof P);
    P.x = (const float*)d_in[0]; P.w = (const float*)d_in[1];
    P.me = (char*)d_ws_all[rank];
    int me = 0;
    while (phys(me) != (unsigned)rank) ++me;
    for (int l = 0; l < 7; ++l) P.lw[l] = (char*)d_ws_all[phys(link_rank(me, l))];
    P.out = (float*)d_out;
    P.rank = me;
    ag_gemm_kernel<<<NWG, 512, LDS_BYTES, stream>>>(P);
    HIP_CHECK(hipGetLastError());
}
```
